```python
import numpy as np

kernel_name = "gpu_a2a_gemm_m8192_k8192_n8192_f32_relu"


ABSMAX_RTOL = 0.001

COMPUTE_DTYPE = "float32"
EPILOGUE = "relu"
IN_QUANT = None
OUT_QUANT = None
IN_DTYPES = {"x": "float32", "w_mat": "float32"}
WORLD_SIZE = 8

def setup_inputs():

    x = np.random.randn(8192, 8192).astype(np.float32)
    w_mat = (np.random.randn(8192, 8192) / np.sqrt(8192)).astype(np.float32)
    return {"x": x, "w_mat": w_mat}

def _epilogue(y):
    return np.maximum(y, 0.0).astype(np.float32)

def reference(x, w_mat):
    y = (x @ w_mat).astype(np.float32)
    return _epilogue(y)
```

```cpp
#include <hip/hip_runtime.h>
#include <rccl/rccl.h>
#include <cstdio>
#include <cstdint>

#define HIP_CHECK(expr) do { hipError_t _e = (expr); if (_e != hipSuccess) { \
    fprintf(stderr, "HIP error %s at %s:%d\n", hipGetErrorString(_e), __FILE__, __LINE__); \
    return; } } while (0)

typedef _Float16 f16;
typedef f16 f16x8 __attribute__((ext_vector_type(8)));
typedef f16 f16x4 __attribute__((ext_vector_type(4)));
typedef float f32x4 __attribute__((ext_vector_type(4)));
typedef short s16x4v __attribute__((__vector_size__(4 * sizeof(short))));
typedef __attribute__((address_space(3))) s16x4v* lds_s4p;

namespace {
constexpr int NR = 8;
constexpr int ML = 1024, NN = 8192, KD = 8192, KL = 1024;
constexpr int BK = 64;
constexpr int KT_ALL = KD / BK;
constexpr int KT_HALF = KT_ALL / 2;
constexpr int IMG_BYTES = 256 * BK * 2;
constexpr int LDS_BYTES = 5 * IMG_BYTES;
constexpr int B_BASE = 3 * IMG_BYTES;
constexpr int GRID = 256;

constexpr size_t WS_TICKET = 0;
constexpr size_t WS_TMO = 128;
constexpr size_t WS_FLAGS = 4096;
constexpr size_t WS_SPLITK = 65536;
constexpr size_t WS_ASTAGE = 1u << 20;
constexpr size_t ASTAGE_BYTES = (size_t)4 * KT_ALL * IMG_BYTES;
constexpr size_t WS_PARTIAL = WS_ASTAGE + 2 * ASTAGE_BYTES;
constexpr size_t WS_TOTAL = WS_PARTIAL + (size_t)128 * 32 * 512 * 16;

constexpr unsigned long long DEADLINE_TICKS = 20000000ull;

struct PeerTable { char* ws[NR]; };

struct OrderTab { unsigned char v[2][32]; };
constexpr OrderTab make_order() {
  OrderTab t{};
  for (int h = 0; h < 2; ++h) {
    int n = 0;
    for (int kc = 4 * h; kc < 4 * h + 4; ++kc) t.v[h][n++] = (unsigned char)kc;
    for (int kc = 0; kc < 8; ++kc)
      for (int sr = 1; sr < 8; ++sr)
        if (((sr + kc) & 1) == h) t.v[h][n++] = (unsigned char)(sr * 8 + kc);
  }
  return t;
}
__constant__ OrderTab g_order = make_order();
}

__device__ __forceinline__ void glds16(const void* g, unsigned lds_off, char* smem) {
  __builtin_amdgcn_global_load_lds((const __attribute__((address_space(1))) void*)g,
                                   (__attribute__((address_space(3))) void*)(smem + lds_off), 16, 0, 0);
}

#define WAIT_V(n) asm volatile("s_waitcnt vmcnt(" #n ")" ::: "memory")
#define WAIT_L(n) asm volatile("s_waitcnt lgkmcnt(" #n ")" ::: "memory")
#define BAR() do { asm volatile("" ::: "memory"); __builtin_amdgcn_s_barrier(); asm volatile("" ::: "memory"); } while (0)
#define SCHED() __builtin_amdgcn_sched_barrier(0)

__device__ __forceinline__ void bounded_wait(const unsigned* flag, unsigned want, unsigned* tmo, unsigned call) {
  const unsigned long long t0 = __builtin_amdgcn_s_memrealtime();
  while (__hip_atomic_load(flag, __ATOMIC_RELAXED, __HIP_MEMORY_SCOPE_SYSTEM) < want) {
    if (__hip_atomic_load(tmo, __ATOMIC_RELAXED, __HIP_MEMORY_SCOPE_AGENT) == call ||
        __builtin_amdgcn_s_memrealtime() - t0 > DEADLINE_TICKS) {
      __hip_atomic_store(tmo, call, __ATOMIC_RELAXED, __HIP_MEMORY_SCOPE_AGENT);
      break;
    }
    __builtin_amdgcn_s_sleep(2);
  }
}

__global__ __launch_bounds__(512, 2)
void a2a_gemm_kernel(const float* __restrict__ X,
                     const float* __restrict__ W,
                     float* __restrict__ out,
                     PeerTable peers, int rank)
{
  extern __shared__ __attribute__((aligned(16))) char smem[];
  char* const ws = peers.ws[rank];
  unsigned* const tmo = (unsigned*)(ws + WS_TMO);
  const int tid = threadIdx.x;
  const int lane = tid & 63;
  const int wid = __builtin_amdgcn_readfirstlane(tid >> 6);
  const int wr = wid >> 2, wc = wid & 3;
  const int fr = lane & 15, fq = lane >> 4;
  const int b = blockIdx.x;

  volatile unsigned* s_bc = (volatile unsigned*)(smem + 2 * IMG_BYTES);
  if (tid == 0) {
    unsigned tk = __hip_atomic_fetch_add((unsigned*)(ws + WS_TICKET), 1u, __ATOMIC_RELAXED, __HIP_MEMORY_SCOPE_AGENT);
    *s_bc = tk / GRID + 1u;
  }
  __syncthreads();
  const unsigned call = __builtin_amdgcn_readfirstlane(*s_bc);
  const unsigned parity = call & 1u;

  {
    const int kc = b >> 5, d_rel = (b >> 2) & 7, pmt = b & 3;
    const int dst = (rank + d_rel) & (NR - 1);
    char* dbase = peers.ws[dst] + WS_ASTAGE + parity * ASTAGE_BYTES +
                  ((size_t)pmt * KT_ALL + (size_t)(rank * 16 + kc * 2)) * IMG_BYTES;
    const float* xs = X + (size_t)(dst * ML + pmt * 256) * KL + kc * 128;
    f32x4 v[8][2];
#pragma unroll
    for (int i = 0; i < 8; ++i) {
      const int c = i * 512 + tid;
      const int img = c >> 11, pos = c & 2047, R = pos >> 3;
      const int ch = (pos & 7) ^ (((R >> 1) & 7) ^ ((R & 1) << 2));
      const float* p = xs + (size_t)R * KL + img * 64 + ch * 8;
      v[i][0] = *(const f32x4*)p;
      v[i][1] = *(const f32x4*)(p + 4);
    }
#pragma unroll
    for (int i = 0; i < 8; ++i) {
      f16x8 h;
#pragma unroll
      for (int e = 0; e < 4; ++e) { h[e] = (f16)v[i][0][e]; h[4 + e] = (f16)v[i][1][e]; }
      *(f16x8*)(dbase + (size_t)(i * 512 + tid) * 16) = h;
    }
    WAIT_V(0);
    __syncthreads();
    if (tid == 0) {
      __builtin_amdgcn_fence(__ATOMIC_RELEASE, "");
      asm volatile("s_waitcnt vmcnt(0)" ::: "memory");
      __hip_atomic_store((unsigned*)(peers.ws[dst] + WS_FLAGS + (size_t)((rank * 8 + kc) * 4 + pmt) * 128), call,
                         __ATOMIC_RELAXED, __HIP_MEMORY_SCOPE_SYSTEM);
    }
  }

  const int xcd = b & 7, idx = b >> 3;
  const int nt = xcd * 4 + (idx & 3);
  const int mt = (idx >> 2) & 3;
  const int kh = idx >> 4;

  const int frs = ((fr >> 1) & 7) ^ ((fr & 1) << 2);
  const unsigned a_rd = (unsigned)((wr * 128 + fr) * 128 + ((fq ^ frs) << 4));
  unsigned b_rd[4];
  {
    const int g = lane >> 4, q = (lane & 15) >> 2, p = lane & 3;
    const int k = 8 * g + q;
    const int h = (k & 1) | (((k >> 1) & 1) << 1) | (((k >> 3) & 1) << 2);
#pragma unroll
    for (int nf = 0; nf < 4; ++nf) b_rd[nf] = (unsigned)(B_BASE + k * 512 + (((wc * 4 + nf) ^ h) * 32) + p * 8);
  }
  const unsigned b_wr0 = (unsigned)(B_BASE + wid * 8 * 512 + (lane & 3) * 8);
  const unsigned b_wr1 = (unsigned)(((lane >> 2) ^ ((wid & 1) << 2)) * 32);
  const unsigned b_voff = (unsigned)(lane * 16);
  const unsigned a_dst = (unsigned)(wid * 4096);

  f32x4 acc[8][4];
#pragma unroll
  for (int m = 0; m < 8; ++m)
#pragma unroll
    for (int n = 0; n < 4; ++n) acc[m][n] = (f32x4){0.f, 0.f, 0.f, 0.f};

  f32x4 br[8];
  f16x8 At[4][2], B0[2][2], B1[2][2];

  const char* a_src = ws + WS_ASTAGE + parity * ASTAGE_BYTES + ((size_t)mt * KT_ALL) * IMG_BYTES + wid * 4096 + lane * 16;
  const char* w_src = (const char*)(W + (size_t)(wid * 8) * NN + (size_t)nt * 256);
  const unsigned* flag_base = (const unsigned*)(ws + WS_FLAGS) + mt * 32;

  auto kt_of = [&](int t) -> int {
    t = t < KT_HALF ? t : KT_HALF - 1;
    const int e = g_order.v[kh][t >> 1];
    return (((rank + (e >> 3)) & (NR - 1)) * 16) + (e & 7) * 2 + (t & 1);
  };
  auto flag_of = [&](int t) -> const unsigned* {
    t = t < KT_HALF ? t : KT_HALF - 1;
    const int e = g_order.v[kh][t >> 1];
    return flag_base + (size_t)(((rank + (e >> 3)) & (NR - 1)) * 8 + (e & 7)) * 128;
  };
  auto need = [&](int t) { bounded_wait(flag_of(t), call, tmo, call); asm volatile("buffer_inv sc0 sc1" ::: "memory"); };

#define ISSUE_A(kt, aoff, q) glds16(a_src + (size_t)(kt) * IMG_BYTES + (q) * 1024, (aoff) + a_dst + (q) * 1024, smem)

#define ISSUE_B(kt, q) do { \
    const char* g_ = w_src + (size_t)(kt) * (BK * (size_t)NN * 4); \
    _Pragma("unroll") for (int j_ = 2 * (q); j_ < 2 * (q) + 2; ++j_) { \
      const char* gj_ = g_ + (size_t)j_ * (NN * 4); \
      asm volatile("global_load_dwordx4 %0, %1, %2" : "=v"(br[j_]) : "v"(b_voff), "s"(gj_)); \
    } } while (0)

#define WAIT_B(q, n) asm volatile("s_waitcnt vmcnt(" #n ")" : "+v"(br[2 * (q)]), "+v"(br[2 * (q) + 1]))

#define WRITE_B(stage, q) do { \
    _Pragma("unroll") for (int j_ = 2 * (q); j_ < 2 * (q) + 2; ++j_) { \
      f16x4 h_; \
      _Pragma("unroll") for (int i_ = 0; i_ < 4; ++i_) h_[i_] = (f16)br[j_][i_]; \
      *(f16x4*)(smem + (stage) * IMG_BYTES + j_ * 512 + b_wr0 + (b_wr1 ^ ((j_ & 3) * 32))) = h_; \
    } } while (0)

#define LDA(mh) do { \
    _Pragma("unroll") for (int m_ = 0; m_ < 4; ++m_) { \
      At[m_][0] = *(const f16x8*)(smem + a_rdc + ((mh) * 4 + m_) * 2048); \
      At[m_][1] = *(const f16x8*)(smem + (a_rdc ^ 64) + ((mh) * 4 + m_) * 2048); \
    } } while (0)
#define TRRD(off) __builtin_bit_cast(f16x4, __builtin_amdgcn_ds_read_tr16_b64_v4i16((lds_s4p)(smem + (off))))
#define LDB(dst, stage, nh) do { \
    _Pragma("unroll") for (int n_ = 0; n_ < 2; ++n_) \
    _Pragma("unroll") for (int k_ = 0; k_ < 2; ++k_) { \
      f16x4 lo_ = TRRD((stage) * IMG_BYTES + b_rd[(nh) * 2 + n_] + k_ * 16384); \
      f16x4 hi_ = TRRD((stage) * IMG_BYTES + b_rd[(nh) * 2 + n_] + k_ * 16384 + 2048); \
      dst[n_][k_] = __builtin_shufflevector(lo_, hi_, 0, 1, 2, 3, 4, 5, 6, 7); \
    } } while (0)
#define MMA(mh, nh, Bf) do { __builtin_amdgcn_s_setprio(1); \
    _Pragma("unroll") for (int m_ = 0; m_ < 4; ++m_) \
    _Pragma("unroll") for (int n_ = 0; n_ < 2; ++n_) \
    _Pragma("unroll") for (int k_ = 0; k_ < 2; ++k_) \
      acc[(mh) * 4 + m_][(nh) * 2 + n_] = __builtin_amdgcn_mfma_f32_16x16x32_f16(Bf[n_][k_], At[m_][k_], acc[(mh) * 4 + m_][(nh) * 2 + n_], 0, 0, 0); \
    __builtin_amdgcn_s_setprio(0); } while (0)

  {
    need(0);
    const int k0 = kt_of(0), k1 = kt_of(1);
    ISSUE_A(k0, 0, 0); ISSUE_A(k0, 0, 1); ISSUE_A(k0, 0, 2); ISSUE_A(k0, 0, 3);
    ISSUE_B(k0, 0); ISSUE_B(k0, 1); ISSUE_B(k0, 2); ISSUE_B(k0, 3);
    WAIT_B(0, 6); WRITE_B(0, 0);
    WAIT_B(1, 4); WRITE_B(0, 1);
    WAIT_B(2, 2); WRITE_B(0, 2);
    WAIT_B(3, 0); WRITE_B(0, 3);
    ISSUE_A(k1, IMG_BYTES, 0); ISSUE_B(k1, 0);
    ISSUE_A(k1, IMG_BYTES, 1); ISSUE_B(k1, 1);
    ISSUE_A(k1, IMG_BYTES, 2); ISSUE_B(k1, 2);
    ISSUE_A(k1, IMG_BYTES, 3); ISSUE_B(k1, 3);
  }
  WAIT_L(0);
  BAR();
  if (wr == 1) BAR();

  unsigned a_rdc = a_rd;
  int a_cur = 0;
#define KTILE(t, s) do { \
    if (((t) & 1) == 0) need((t) + 2); \
    const int ktn2 = kt_of((t) + 2); \
    const unsigned a_nx = (unsigned)(a_cur == 0 ? 2 : a_cur - 1) * IMG_BYTES; \
      \
    LDB(B0, s, 0); SCHED(); LDA(0); ISSUE_A(ktn2, a_nx, 0); \
    WAIT_B(0, 10); WRITE_B((s) ^ 1, 0); ISSUE_B(ktn2, 0); \
    BAR(); WAIT_L(0); MMA(0, 0, B0); BAR(); SCHED(); \
      \
    LDB(B1, s, 1); ISSUE_A(ktn2, a_nx, 1); \
    WAIT_B(1, 10); WRITE_B((s) ^ 1, 1); ISSUE_B(ktn2, 1); \
    BAR(); WAIT_L(0); MMA(0, 1, B1); BAR(); SCHED(); \
      \
    LDA(1); ISSUE_A(ktn2, a_nx, 2); \
    WAIT_B(2, 10); WRITE_B((s) ^ 1, 2); ISSUE_B(ktn2, 2); \
    BAR(); WAIT_L(0); MMA(1, 1, B1); BAR(); SCHED(); \
      \
    ISSUE_A(ktn2, a_nx, 3); \
    WAIT_B(3, 10); WRITE_B((s) ^ 1, 3); ISSUE_B(ktn2, 3); WAIT_L(0); \
    BAR(); MMA(1, 0, B0); \
    a_cur = (a_cur == 2) ? 0 : a_cur + 1; a_rdc = a_rd + (unsigned)a_cur * IMG_BYTES; \
    BAR(); SCHED(); \
  } while (0)

  for (int t = 0; t < KT_HALF; t += 2) {
    KTILE(t, 0);
    KTILE(t + 1, 1);
  }
#undef KTILE
  if (wr == 0) BAR();
  WAIT_V(0);
  BAR();

  int tz = threadIdx.x; asm volatile("" : "+v"(tz));
  const int lane2 = tz & 63;
  const int fr2 = lane2 & 15, fq2 = lane2 >> 4;
  const int tile = nt * 4 + mt;
  unsigned* const sk = (unsigned*)(ws + WS_SPLITK + (size_t)tile * 256);
  volatile unsigned* s_tk = (volatile unsigned*)smem;
  if (tz == 0) *s_tk = __hip_atomic_fetch_add(sk, 1u, __ATOMIC_RELAXED, __HIP_MEMORY_SCOPE_AGENT);
  __syncthreads();
  const unsigned ticket = __builtin_amdgcn_readfirstlane(*s_tk);
  f32x4* pp = (f32x4*)(ws + WS_PARTIAL) + (size_t)tile * (32 * 512) + tz;
  if ((ticket & 1u) == 0u) {
#pragma unroll
    for (int m = 0; m < 8; ++m)
#pragma unroll
      for (int n = 0; n < 4; ++n) pp[(m * 4 + n) * 512] = acc[m][n];
    WAIT_V(0);
    __syncthreads();
    if (tz == 0) {
      __builtin_amdgcn_fence(__ATOMIC_RELEASE, "agent");
      asm volatile("s_waitcnt vmcnt(0)" ::: "memory");
      __hip_atomic_store(sk + 32, call, __ATOMIC_RELAXED, __HIP_MEMORY_SCOPE_AGENT);
    }
    return;
  }
  if (tz == 0) {
    bounded_wait(sk + 32, call, tmo, call);
    __builtin_amdgcn_fence(__ATOMIC_ACQUIRE, "agent");
    asm volatile("s_waitcnt vmcnt(0)" ::: "memory");
    *s_tk = (__hip_atomic_load(tmo, __ATOMIC_RELAXED, __HIP_MEMORY_SCOPE_AGENT) == call) ? 1u : 0u;
  }
  __syncthreads();
  const bool poisoned = __builtin_amdgcn_readfirstlane(*s_tk) != 0u;
  float* orow = out + (size_t)(mt * 256 + wr * 128 + fr2) * NN + nt * 256 + wc * 64 + fq2 * 4;
  if (poisoned) {
    const float qn = __builtin_nanf("");
#pragma unroll
    for (int m = 0; m < 8; ++m)
#pragma unroll
      for (int n = 0; n < 4; ++n) *(f32x4*)(orow + (size_t)m * 16 * NN + n * 16) = (f32x4){qn, qn, qn, qn};
    return;
  }
#pragma unroll
  for (int m = 0; m < 8; ++m)
#pragma unroll
    for (int n = 0; n < 4; ++n) {
      f32x4 v = acc[m][n] + pp[(m * 4 + n) * 512];
      v[0] = fmaxf(v[0], 0.f); v[1] = fmaxf(v[1], 0.f); v[2] = fmaxf(v[2], 0.f); v[3] = fmaxf(v[3], 0.f);
      *(f32x4*)(orow + (size_t)m * 16 * NN + n * 16) = v;
    }
}

extern "C" void kernel_launch(int rank, int nranks, ncclComm_t comm,
                              hipStream_t stream,
                              void* const* d_in, const int* in_sizes, int n_in,
                              void* d_out, int out_size,
                              const int* in_dtypes, int out_dtype,
                              void* const* d_out_all,
                              void* const* d_ws_all, size_t ws_bytes) {
  (void)comm; (void)in_sizes; (void)n_in; (void)out_size; (void)in_dtypes; (void)out_dtype; (void)d_out_all;
  if (nranks != NR || (unsigned)rank >= (unsigned)NR || ws_bytes < WS_TOTAL) {
    fprintf(stderr, "a2a_gemm: unexpected configuration\n");
    return;
  }
  static bool inited = false;
  if (!inited) {
    HIP_CHECK(hipFuncSetAttribute((const void*)a2a_gemm_kernel, hipFuncAttributeMaxDynamicSharedMemorySize, LDS_BYTES));
    inited = true;
  }
  PeerTable pt;
  for (int p = 0; p < NR; ++p) pt.ws[p] = (char*)d_ws_all[p];
  hipLaunchKernelGGL(a2a_gemm_kernel, dim3(GRID), dim3(512), LDS_BYTES, stream,
                     (const float*)d_in[0], (const float*)d_in[1], (float*)d_out, pt, rank);
  HIP_CHECK(hipGetLastError());
}
```

```cpp
#include <hip/hip_runtime.h>
#include <rccl/rccl.h>
#include <cstdio>
#include <cstdint>

#define HIP_CHECK(expr) do { hipError_t _e = (expr); if (_e != hipSuccess) { \
    fprintf(stderr, "HIP error %s at %s:%d\n", hipGetErrorString(_e), __FILE__, __LINE__); \
    return; } } while (0)

typedef _Float16 f16;
typedef f16 f16x8 __attribute__((ext_vector_type(8)));
typedef f16 f16x4 __attribute__((ext_vector_type(4)));
typedef float f32x4 __attribute__((ext_vector_type(4)));
typedef short s16x4v __attribute__((__vector_size__(4 * sizeof(short))));
typedef __attribute__((address_space(3))) s16x4v* lds_s4p;

namespace {
constexpr int NR = 8;
constexpr int ML = 1024, NN = 8192, KD = 8192, KL = 1024;
constexpr int BK = 64;
constexpr int KT_ALL = KD / BK;
constexpr int KT_HALF = KT_ALL / 2;
constexpr int IMG_BYTES = 256 * BK * 2;
constexpr int LDS_BYTES = 5 * IMG_BYTES;
constexpr int B_BASE = 3 * IMG_BYTES;
constexpr int GRID = 256;

constexpr size_t WS_TICKET = 0;
constexpr size_t WS_TMO = 128;
constexpr size_t WS_FLAGS = 4096;
constexpr size_t WS_SPLITK = 65536;
constexpr size_t WS_DBG = 524288;
constexpr size_t WS_ASTAGE = 1u << 20;
constexpr size_t ASTAGE_BYTES = (size_t)4 * KT_ALL * IMG_BYTES;
constexpr size_t WS_PARTIAL = WS_ASTAGE + 2 * ASTAGE_BYTES;
constexpr size_t WS_TOTAL = WS_PARTIAL + (size_t)128 * 32 * 512 * 16;

constexpr unsigned long long DEADLINE_TICKS = 20000000ull;

struct PeerTable { char* ws[NR]; };

struct OrderTab { unsigned char v[2][32]; };
constexpr OrderTab make_order() {
  OrderTab t{};
  for (int h = 0; h < 2; ++h) {
    int n = 0;
    for (int kc = 4 * h; kc < 4 * h + 4; ++kc) t.v[h][n++] = (unsigned char)kc;
    for (int kc = 0; kc < 8; ++kc)
      for (int sr = 1; sr < 8; ++sr)
        if (((sr + kc) & 1) == h) t.v[h][n++] = (unsigned char)(sr * 8 + kc);
  }
  return t;
}
__constant__ OrderTab g_order = make_order();
}

__device__ __forceinline__ void glds16(const void* g, unsigned lds_off, char* smem) {
  __builtin_amdgcn_global_load_lds((const __attribute__((address_space(1))) void*)g,
                                   (__attribute__((address_space(3))) void*)(smem + lds_off), 16, 0, 0);
}

#define WAIT_V(n) asm volatile("s_waitcnt vmcnt(" #n ")" ::: "memory")
#define WAIT_L(n) asm volatile("s_waitcnt lgkmcnt(" #n ")" ::: "memory")
#define BAR() do { asm volatile("" ::: "memory"); __builtin_amdgcn_s_barrier(); asm volatile("" ::: "memory"); } while (0)
#define SCHED() __builtin_amdgcn_sched_barrier(0)

__device__ __forceinline__ void bounded_wait(const unsigned* flag, unsigned want, unsigned* tmo, unsigned call) {
  const unsigned long long t0 = __builtin_amdgcn_s_memrealtime();
  while (__hip_atomic_load(flag, __ATOMIC_RELAXED, __HIP_MEMORY_SCOPE_SYSTEM) < want) {
    if (__hip_atomic_load(tmo, __ATOMIC_RELAXED, __HIP_MEMORY_SCOPE_AGENT) == call ||
        __builtin_amdgcn_s_memrealtime() - t0 > DEADLINE_TICKS) {
      __hip_atomic_store(tmo, call, __ATOMIC_RELAXED, __HIP_MEMORY_SCOPE_AGENT);
      break;
    }
    __builtin_amdgcn_s_sleep(2);
  }
}

__device__ __forceinline__ const char* uni_ptr(const void* p) {
  const unsigned long long v = (unsigned long long)p;
  unsigned lo = __builtin_amdgcn_readfirstlane((unsigned)v), hi = __builtin_amdgcn_readfirstlane((unsigned)(v >> 32));
  asm volatile("s_nop 4" : "+s"(lo), "+s"(hi));
  return (const char*)(((unsigned long long)hi << 32) | lo);
}

__global__ __launch_bounds__(512, 2)
void a2a_gemm_kernel(const float* __restrict__ X,
                     const float* __restrict__ W,
                     float* __restrict__ out,
                     PeerTable peers, int rank)
{
  extern __shared__ __attribute__((aligned(16))) char smem[];
  char* const ws = peers.ws[rank];
#define STAMP(i) do {} while (0)
  STAMP(0);
  unsigned* const tmo = (unsigned*)(ws + WS_TMO);
  const int tid = threadIdx.x;
  const int lane = tid & 63;
  const int wid = __builtin_amdgcn_readfirstlane(tid >> 6);
  const int wr = wid >> 2, wc = wid & 3;
  const int fr = lane & 15, fq = lane >> 4;
  const int b = blockIdx.x;

  volatile unsigned* s_bc = (volatile unsigned*)(smem + 2 * IMG_BYTES);
  if (tid == 0) {
    unsigned tk = __hip_atomic_fetch_add((unsigned*)(ws + WS_TICKET), 1u, __ATOMIC_RELAXED, __HIP_MEMORY_SCOPE_AGENT);
    *s_bc = tk / GRID + 1u;
  }
  __syncthreads();
  const unsigned call = __builtin_amdgcn_readfirstlane(*s_bc);
  const unsigned parity = call & 1u;

  const int p_kc = b >> 5, p_drel = (b >> 2) & 7, p_mt = b & 3;
  const int p_dst = (rank + p_drel) & (NR - 1);
  const int t_push = (p_drel == 0) ? -1 : 4 * p_kc;
  auto do_push = [&]() {
    const char* dbase = uni_ptr(peers.ws[p_dst] + WS_ASTAGE + parity * ASTAGE_BYTES +
                                ((size_t)p_mt * KT_ALL + (size_t)(rank * 16 + p_kc * 2)) * IMG_BYTES);
    const float* xs = X + (size_t)(p_dst * ML + p_mt * 256) * KL + p_kc * 128;
    int tq = threadIdx.x; asm volatile("" : "+v"(tq));
    f32x4 v[8][2];
#pragma unroll
    for (int i = 0; i < 8; ++i) {
      const int c = i * 512 + tq;
      const int img = c >> 11, pos = c & 2047, R = pos >> 3;
      const int ch = (pos & 7) ^ (((R >> 1) & 7) ^ ((R & 1) << 2));
      const float* p = xs + (size_t)R * KL + img * 64 + ch * 8;
      v[i][0] = *(const f32x4*)p;
      v[i][1] = *(const f32x4*)(p + 4);
    }
    const unsigned voff = (unsigned)tq * 16u;
#pragma unroll
    for (int i = 0; i < 8; ++i) {
      f16x8 h;
#pragma unroll
      for (int e = 0; e < 4; ++e) { h[e] = (f16)v[i][0][e]; h[4 + e] = (f16)v[i][1][e]; }
      const char* db = dbase + (size_t)i * 8192;
      asm volatile("global_store_dwordx4 %0, %1, %2 sc0 sc1\n\ts_nop 1" :: "v"(voff), "v"(h), "s"(db) : "memory");
    }
  };
  auto signal_push = [&]() {
    if (tid == 0)
      __hip_atomic_store((unsigned*)(peers.ws[p_dst] + WS_FLAGS + (size_t)((rank * 8 + p_kc) * 4 + p_mt) * 128), call,
                         __ATOMIC_RELAXED, __HIP_MEMORY_SCOPE_SYSTEM);
  };
  if (t_push < 0) {
    do_push();
    WAIT_V(0);
    __syncthreads();
    signal_push();
  }
  STAMP(1);

  const int xcd = b & 7, idx = b >> 3;
  const int nt = xcd * 4 + (idx & 3);
  const int mt = (idx >> 2) & 3;
  const int kh = idx >> 4;

  const int frs = ((fr >> 1) & 7) ^ ((fr & 1) << 2);
  const unsigned a_rd = (unsigned)((wr * 128 + fr) * 128 + ((fq ^ frs) << 4));
  unsigned b_rd[4];
  {
    const int g = lane >> 4, q = (lane & 15) >> 2, p = lane & 3;
    const int k = 8 * g + q;
    const int h = (k & 1) | (((k >> 1) & 1) << 1) | (((k >> 3) & 1) << 2);
#pragma unroll
    for (int nf = 0; nf < 4; ++nf) b_rd[nf] = (unsigned)(B_BASE + k * 512 + (((wc * 4 + nf) ^ h) * 32) + p * 8);
  }
  const unsigned b_wr0 = (unsigned)(B_BASE + wid * 8 * 512 + (lane & 3) * 8);
  const unsigned b_wr1 = (unsigned)(((lane >> 2) ^ ((wid & 1) << 2)) * 32);
  const unsigned b_voff = (unsigned)(lane * 16);
  const unsigned a_dst = (unsigned)(wid * 4096);

  f32x4 acc[8][4];
#pragma unroll
  for (int m = 0; m < 8; ++m)
#pragma unroll
    for (int n = 0; n < 4; ++n) acc[m][n] = (f32x4){0.f, 0.f, 0.f, 0.f};

  f32x4 br[8];
  f16x8 At[4][2], B0[2][2], B1[2][2];

  const char* a_src = ws + WS_ASTAGE + parity * ASTAGE_BYTES + ((size_t)mt * KT_ALL) * IMG_BYTES + wid * 4096 + lane * 16;
  const char* w_src = (const char*)(W + (size_t)(wid * 8) * NN + (size_t)nt * 256);
  const unsigned* flag_base = (const unsigned*)(ws + WS_FLAGS) + mt * 32;

  auto kt_of = [&](int t) -> int {
    t = t < KT_HALF ? t : KT_HALF - 1;
    const int e = __builtin_amdgcn_readfirstlane((int)g_order.v[kh][t >> 1]);
    return (((rank + (e >> 3)) & (NR - 1)) * 16) + (e & 7) * 2 + (t & 1);
  };
  auto flag_of = [&](int t) -> const unsigned* {
    t = t < KT_HALF ? t : KT_HALF - 1;
    const int e = __builtin_amdgcn_readfirstlane((int)g_order.v[kh][t >> 1]);
    return flag_base + (size_t)(((rank + (e >> 3)) & (NR - 1)) * 8 + (e & 7)) * 128;
  };
  unsigned flv = 0;
  const unsigned vzero = 0;
#define FLAG_FETCH(t) do { const char* fp_ = uni_ptr(flag_of(t)); \
    asm volatile("global_load_dword %0, %1, %2 sc0 sc1" : "=v"(flv) : "v"(vzero), "s"(fp_)); } while (0)
#define FLAG_CHECK(t) do { asm volatile("" : "+v"(flv) :: "memory"); \
    if (__builtin_amdgcn_readfirstlane(flv) < call) bounded_wait(flag_of(t), call, tmo, call); } while (0)

#define ISSUE_A(kt, aoff, q) glds16(a_src + (size_t)(kt) * IMG_BYTES + (q) * 1024, (aoff) + a_dst + (q) * 1024, smem)

#define ISSUE_B(kt, q) do { \
    const char* g_ = uni_ptr(w_src + (size_t)(kt) * (BK * (size_t)NN * 4)); \
    _Pragma("unroll") for (int j_ = 2 * (q); j_ < 2 * (q) + 2; ++j_) { \
      const char* gj_ = g_ + (size_t)j_ * (NN * 4); \
      asm volatile("global_load_dwordx4 %0, %1, %2" : "=v"(br[j_]) : "v"(b_voff), "s"(gj_)); \
    } } while (0)

#define WAIT_B(q, n) asm volatile("s_waitcnt vmcnt(" #n ")" : "+v"(br[2 * (q)]), "+v"(br[2 * (q) + 1]))

#define WRITE_B(stage, q) do { \
    _Pragma("unroll") for (int j_ = 2 * (q); j_ < 2 * (q) + 2; ++j_) { \
      f16x4 h_; \
      _Pragma("unroll") for (int i_ = 0; i_ < 4; ++i_) h_[i_] = (f16)br[j_][i_]; \
      *(f16x4*)(smem + (stage) * IMG_BYTES + j_ * 512 + b_wr0 + (b_wr1 ^ ((j_ & 3) * 32))) = h_; \
    } } while (0)

#define LDA(mh) do { \
    _Pragma("unroll") for (int m_ = 0; m_ < 4; ++m_) { \
      At[m_][0] = *(const f16x8*)(smem + a_rdc + ((mh) * 4 + m_) * 2048); \
      At[m_][1] = *(const f16x8*)(smem + (a_rdc ^ 64) + ((mh) * 4 + m_) * 2048); \
    } } while (0)
#define TRRD(off) __builtin_bit_cast(f16x4, __builtin_amdgcn_ds_read_tr16_b64_v4i16((lds_s4p)(smem + (off))))
#define LDB(dst, stage, nh) do { \
    _Pragma("unroll") for (int n_ = 0; n_ < 2; ++n_) \
    _Pragma("unroll") for (int k_ = 0; k_ < 2; ++k_) { \
      f16x4 lo_ = TRRD((stage) * IMG_BYTES + b_rd[(nh) * 2 + n_] + k_ * 16384); \
      f16x4 hi_ = TRRD((stage) * IMG_BYTES + b_rd[(nh) * 2 + n_] + k_ * 16384 + 2048); \
      dst[n_][k_] = __builtin_shufflevector(lo_, hi_, 0, 1, 2, 3, 4, 5, 6, 7); \
    } } while (0)
#define MMA(mh, nh, Bf) do { __builtin_amdgcn_s_setprio(1); \
    _Pragma("unroll") for (int m_ = 0; m_ < 4; ++m_) \
    _Pragma("unroll") for (int n_ = 0; n_ < 2; ++n_) \
    _Pragma("unroll") for (int k_ = 0; k_ < 2; ++k_) \
      acc[(mh) * 4 + m_][(nh) * 2 + n_] = __builtin_amdgcn_mfma_f32_16x16x32_f16(Bf[n_][k_], At[m_][k_], acc[(mh) * 4 + m_][(nh) * 2 + n_], 0, 0, 0); \
    __builtin_amdgcn_s_setprio(0); } while (0)

  {
    bounded_wait(flag_of(0), call, tmo, call);
    FLAG_FETCH(2);
    const int k0 = kt_of(0), k1 = kt_of(1);
    ISSUE_A(k0, 0, 0); ISSUE_A(k0, 0, 1); ISSUE_A(k0, 0, 2); ISSUE_A(k0, 0, 3);
    ISSUE_B(k0, 0); ISSUE_B(k0, 1); ISSUE_B(k0, 2); ISSUE_B(k0, 3);
    WAIT_B(0, 6); WRITE_B(0, 0);
    WAIT_B(1, 4); WRITE_B(0, 1);
    WAIT_B(2, 2); WRITE_B(0, 2);
    WAIT_B(3, 0); WRITE_B(0, 3);
    ISSUE_A(k1, IMG_BYTES, 0); ISSUE_B(k1, 0);
    ISSUE_A(k1, IMG_BYTES, 1); ISSUE_B(k1, 1);
    ISSUE_A(k1, IMG_BYTES, 2); ISSUE_B(k1, 2);
    ISSUE_A(k1, IMG_BYTES, 3); ISSUE_B(k1, 3);
  }
  WAIT_L(0);
  BAR();
  STAMP(2);
  if (wr == 1) BAR();

  unsigned a_rdc = a_rd;
  int a_cur = 0;
#define KTILE(t, s) do { \
    if (((t) & 1) == 0) { \
      if ((t) == t_push) do_push(); \
      if ((t) == t_push + 2) signal_push(); \
      FLAG_CHECK((t) + 2); \
      FLAG_FETCH((t) + 4); \
    } \
    const int ktn2 = kt_of((t) + 2); \
    const unsigned a_nx = (unsigned)(a_cur == 0 ? 2 : a_cur - 1) * IMG_BYTES; \
      \
    LDB(B0, s, 0); SCHED(); LDA(0); ISSUE_A(ktn2, a_nx, 0); \
    WAIT_B(0, 10); WRITE_B((s) ^ 1, 0); ISSUE_B(ktn2, 0); \
    BAR(); WAIT_L(0); MMA(0, 0, B0); BAR(); SCHED(); \
      \
    LDB(B1, s, 1); ISSUE_A(ktn2, a_nx, 1); \
    WAIT_B(1, 10); WRITE_B((s) ^ 1, 1); ISSUE_B(ktn2, 1); \
    BAR(); WAIT_L(0); MMA(0, 1, B1); BAR(); SCHED(); \
      \
    LDA(1); ISSUE_A(ktn2, a_nx, 2); \
    WAIT_B(2, 10); WRITE_B((s) ^ 1, 2); ISSUE_B(ktn2, 2); \
    BAR(); WAIT_L(0); MMA(1, 1, B1); BAR(); SCHED(); \
      \
    ISSUE_A(ktn2, a_nx, 3); \
    WAIT_B(3, 10); WRITE_B((s) ^ 1, 3); ISSUE_B(ktn2, 3); WAIT_L(0); \
    BAR(); MMA(1, 0, B0); \
    a_cur = (a_cur == 2) ? 0 : a_cur + 1; a_rdc = a_rd + (unsigned)a_cur * IMG_BYTES; \
    BAR(); SCHED(); \
  } while (0)

  for (int t = 0; t < KT_HALF; t += 2) {
    KTILE(t, 0);
    KTILE(t + 1, 1);
  }
#undef KTILE
  if (wr == 0) BAR();
  WAIT_V(0);
  BAR();
  STAMP(3);

  int tz = threadIdx.x; asm volatile("" : "+v"(tz));
  const int lane2 = tz & 63;
  const int fr2 = lane2 & 15, fq2 = lane2 >> 4;
  const int tile = nt * 4 + mt;
  unsigned* const sk = (unsigned*)(ws + WS_SPLITK + (size_t)tile * 256);
  volatile unsigned* s_tk = (volatile unsigned*)smem;
  if (tz == 0) *s_tk = __hip_atomic_fetch_add(sk, 1u, __ATOMIC_RELAXED, __HIP_MEMORY_SCOPE_AGENT);
  __syncthreads();
  const unsigned ticket = __builtin_amdgcn_readfirstlane(*s_tk);
  f32x4* pp = (f32x4*)(ws + WS_PARTIAL) + (size_t)tile * (32 * 512) + tz;
  if ((ticket & 1u) == 0u) {
    {
      const char* pb = uni_ptr((f32x4*)(ws + WS_PARTIAL) + (size_t)tile * (32 * 512));
      const unsigned voff = (unsigned)tz * 16u;
#pragma unroll
      for (int m = 0; m < 8; ++m)
#pragma unroll
        for (int n = 0; n < 4; ++n) {
          const char* pbi = pb + (size_t)(m * 4 + n) * 8192;
          asm volatile("global_store_dwordx4 %0, %1, %2 sc1\n\ts_nop 1" :: "v"(voff), "v"(acc[m][n]), "s"(pbi) : "memory");
        }
    }
    WAIT_V(0);
    __syncthreads();
    if (tz == 0) __hip_atomic_store(sk + 32, call, __ATOMIC_RELAXED, __HIP_MEMORY_SCOPE_AGENT);
    STAMP(4);
    return;
  }
  if (tz == 0) {
    bounded_wait(sk + 32, call, tmo, call);
    __builtin_amdgcn_fence(__ATOMIC_ACQUIRE, "agent");
    asm volatile("s_waitcnt vmcnt(0)" ::: "memory");
    *s_tk = (__hip_atomic_load(tmo, __ATOMIC_RELAXED, __HIP_MEMORY_SCOPE_AGENT) == call) ? 1u : 0u;
  }
  __syncthreads();
  const bool poisoned = __builtin_amdgcn_readfirstlane(*s_tk) != 0u;
  STAMP(5);
  float* orow = out + (size_t)(mt * 256 + wr * 128 + fr2) * NN + nt * 256 + wc * 64 + fq2 * 4;
  if (poisoned) {
    const float qn = __builtin_nanf("");
#pragma unroll
    for (int m = 0; m < 8; ++m)
#pragma unroll
      for (int n = 0; n < 4; ++n) *(f32x4*)(orow + (size_t)m * 16 * NN + n * 16) = (f32x4){qn, qn, qn, qn};
    return;
  }
#pragma unroll
  for (int m = 0; m < 8; ++m)
#pragma unroll
    for (int n = 0; n < 4; ++n) {
      f32x4 v = acc[m][n] + pp[(m * 4 + n) * 512];
      v[0] = fmaxf(v[0], 0.f); v[1] = fmaxf(v[1], 0.f); v[2] = fmaxf(v[2], 0.f); v[3] = fmaxf(v[3], 0.f);
      *(f32x4*)(orow + (size_t)m * 16 * NN + n * 16) = v;
    }
}

extern "C" void kernel_launch(int rank, int nranks, ncclComm_t comm,
                              hipStream_t stream,
                              void* const* d_in, const int* in_sizes, int n_in,
                              void* d_out, int out_size,
                              const int* in_dtypes, int out_dtype,
                              void* const* d_out_all,
                              void* const* d_ws_all, size_t ws_bytes) {
  (void)comm; (void)in_sizes; (void)n_in; (void)out_size; (void)in_dtypes; (void)out_dtype; (void)d_out_all;
  if (nranks != NR || (unsigned)rank >= (unsigned)NR || ws_bytes < WS_TOTAL) {
    fprintf(stderr, "a2a_gemm: unexpected configuration\n");
    return;
  }
  static bool inited = false;
  if (!inited) {
    HIP_CHECK(hipFuncSetAttribute((const void*)a2a_gemm_kernel, hipFuncAttributeMaxDynamicSharedMemorySize, LDS_BYTES));
    inited = true;
  }
  PeerTable pt;
  for (int p = 0; p < NR; ++p) pt.ws[p] = (char*)d_ws_all[p];
  hipLaunchKernelGGL(a2a_gemm_kernel, dim3(GRID), dim3(512), LDS_BYTES, stream,
                     (const float*)d_in[0], (const float*)d_in[1], (float*)d_out, pt, rank);
  HIP_CHECK(hipGetLastError());
}
```

```cpp
#include <hip/hip_runtime.h>
#include <rccl/rccl.h>
#include <cstdio>
#include <cstdint>

#define HIP_CHECK(expr) do { hipError_t _e = (expr); if (_e != hipSuccess) { \
    fprintf(stderr, "HIP error %s at %s:%d\n", hipGetErrorString(_e), __FILE__, __LINE__); \
    return; } } while (0)

typedef _Float16 f16;
typedef f16 f16x8 __attribute__((ext_vector_type(8)));
typedef f16 f16x4 __attribute__((ext_vector_type(4)));
typedef float f32x4 __attribute__((ext_vector_type(4)));
typedef short s16x4v __attribute__((__vector_size__(4 * sizeof(short))));
typedef __attribute__((address_space(3))) s16x4v* lds_s4p;

namespace {
constexpr int NR = 8;
constexpr int ML = 1024, NN = 8192, KD = 8192, KL = 1024;
constexpr int BK = 64;
constexpr int KT_ALL = KD / BK;
constexpr int KT_HALF = KT_ALL / 2;
constexpr int IMG_BYTES = 256 * BK * 2;
constexpr int LDS_BYTES = 5 * IMG_BYTES;
constexpr int B_BASE = 3 * IMG_BYTES;
constexpr int GRID = 256;

constexpr size_t WS_TICKET = 0;
constexpr size_t WS_TMO = 128;
constexpr size_t WS_FLAGS = 4096;
constexpr size_t WS_SPLITK = 65536;
constexpr size_t WS_DBG = 524288;
constexpr size_t WS_ASTAGE = 1u << 20;
constexpr size_t ASTAGE_BYTES = (size_t)4 * KT_ALL * IMG_BYTES;
constexpr size_t WS_PARTIAL = WS_ASTAGE + 2 * ASTAGE_BYTES;
constexpr size_t WS_TOTAL = WS_PARTIAL + (size_t)128 * 32 * 512 * 16;

constexpr unsigned long long DEADLINE_TICKS = 20000000ull;

struct PeerTable { char* ws[NR]; };

struct OrderTab { unsigned char v[2][32]; };
constexpr OrderTab make_order() {
  OrderTab t{};
  for (int h = 0; h < 2; ++h) {
    int n = 0;
    for (int kc = 4 * h; kc < 4 * h + 4; ++kc) t.v[h][n++] = (unsigned char)kc;
    for (int kc = 0; kc < 8; ++kc)
      for (int sr = 1; sr < 8; ++sr)
        if (((sr + kc) & 1) == h) t.v[h][n++] = (unsigned char)(sr * 8 + kc);
  }
  return t;
}
__constant__ OrderTab g_order = make_order();
}

__device__ __forceinline__ void glds16(const void* gsrc, unsigned lds_dst) {
  unsigned keep;
  asm volatile("s_mov_b32 %0, m0\n\ts_mov_b32 m0, %2\n\ts_nop 0\n\tglobal_load_lds_dwordx4 %1, off\n\ts_mov_b32 m0, %0"
               : "=&s"(keep) : "v"(gsrc), "s"(lds_dst) : "memory");
}

#define WAIT_V(n) asm volatile("s_waitcnt vmcnt(" #n ")" ::: "memory")
#define WAIT_L(n) asm volatile("s_waitcnt lgkmcnt(" #n ")" ::: "memory")
#define BAR() do { asm volatile("" ::: "memory"); __builtin_amdgcn_s_barrier(); asm volatile("" ::: "memory"); } while (0)
#define SCHED() __builtin_amdgcn_sched_barrier(0)

__device__ __forceinline__ void bounded_wait(const unsigned* flag, unsigned want, unsigned* tmo, unsigned call) {
  const unsigned long long t0 = __builtin_amdgcn_s_memrealtime();
  while (__hip_atomic_load(flag, __ATOMIC_RELAXED, __HIP_MEMORY_SCOPE_SYSTEM) < want) {
    if (__hip_atomic_load(tmo, __ATOMIC_RELAXED, __HIP_MEMORY_SCOPE_AGENT) == call ||
        __builtin_amdgcn_s_memrealtime() - t0 > DEADLINE_TICKS) {
      __hip_atomic_store(tmo, call, __ATOMIC_RELAXED, __HIP_MEMORY_SCOPE_AGENT);
      break;
    }
    __builtin_amdgcn_s_sleep(2);
  }
}

__device__ __forceinline__ const char* uni_ptr(const void* p) {
  const unsigned long long v = (unsigned long long)p;
  unsigned lo = __builtin_amdgcn_readfirstlane((unsigned)v), hi = __builtin_amdgcn_readfirstlane((unsigned)(v >> 32));
  asm volatile("s_nop 4" : "+s"(lo), "+s"(hi));
  return (const char*)(((unsigned long long)hi << 32) | lo);
}

__global__ __launch_bounds__(512, 2)
void a2a_gemm_kernel(const float* __restrict__ X,
                     const float* __restrict__ W,
                     float* __restrict__ out,
                     PeerTable peers, int rank)
{
  extern __shared__ __attribute__((aligned(16))) char smem[];
  char* const ws = peers.ws[rank];
#define STAMP(i) do {} while (0)
  STAMP(0);
  unsigned* const tmo = (unsigned*)(ws + WS_TMO);
  const int tid = threadIdx.x;
  const int lane = tid & 63;
  const int wid = __builtin_amdgcn_readfirstlane(tid >> 6);
  const int wr = wid >> 2, wc = wid & 3;
  const int fr = lane & 15, fq = lane >> 4;
  const int b = blockIdx.x;

  volatile unsigned* s_bc = (volatile unsigned*)(smem + 2 * IMG_BYTES);
  if (tid == 0) {
    unsigned tk = __hip_atomic_fetch_add((unsigned*)(ws + WS_TICKET), 1u, __ATOMIC_RELAXED, __HIP_MEMORY_SCOPE_AGENT);
    *s_bc = tk / GRID + 1u;
  }
  __syncthreads();
  const unsigned call = __builtin_amdgcn_readfirstlane(*s_bc);
  const unsigned parity = call & 1u;

  const int p_kc = b >> 5, p_drel = (b >> 2) & 7, p_mt = b & 3;
  const int p_dst = (rank + p_drel) & (NR - 1);
  const int t_push = (p_drel == 0) ? -1 : 4 * p_kc;
  auto do_push = [&]() {
    const char* dbase = uni_ptr(peers.ws[p_dst] + WS_ASTAGE + parity * ASTAGE_BYTES +
                                ((size_t)p_mt * KT_ALL + (size_t)(rank * 16 + p_kc * 2)) * IMG_BYTES);
    const float* xs = X + (size_t)(p_dst * ML + p_mt * 256) * KL + p_kc * 128;
    int tq = threadIdx.x; asm volatile("" : "+v"(tq));
    f32x4 v[8][2];
#pragma unroll
    for (int i = 0; i < 8; ++i) {
      const int c = i * 512 + tq;
      const int img = c >> 11, pos = c & 2047, R = pos >> 3;
      const int ch = (pos & 7) ^ (((R >> 1) & 7) ^ ((R & 1) << 2));
      const float* p = xs + (size_t)R * KL + img * 64 + ch * 8;
      v[i][0] = *(const f32x4*)p;
      v[i][1] = *(const f32x4*)(p + 4);
    }
    const unsigned voff = (unsigned)tq * 16u;
#pragma unroll
    for (int i = 0; i < 8; ++i) {
      f16x8 h;
#pragma unroll
      for (int e = 0; e < 4; ++e) { h[e] = (f16)v[i][0][e]; h[4 + e] = (f16)v[i][1][e]; }
      const char* db = dbase + (size_t)i * 8192;
      asm volatile("global_store_dwordx4 %0, %1, %2 sc0 sc1\n\ts_nop 1" :: "v"(voff), "v"(h), "s"(db) : "memory");
    }
  };
  auto signal_push = [&]() {
    if (tid == 0)
      __hip_atomic_store((unsigned*)(peers.ws[p_dst] + WS_FLAGS + (size_t)((rank * 8 + p_kc) * 4 + p_mt) * 128), call,
                         __ATOMIC_RELAXED, __HIP_MEMORY_SCOPE_SYSTEM);
  };
  if (t_push < 0) {
    do_push();
    WAIT_V(0);
    __syncthreads();
    signal_push();
  }
  STAMP(1);

  const int xcd = b & 7, idx = b >> 3;
  const int nt = xcd * 4 + (idx & 3);
  const int mt = (idx >> 2) & 3;
  const int kh = idx >> 4;

  const int frs = ((fr >> 1) & 7) ^ ((fr & 1) << 2);
  const unsigned a_rd = (unsigned)((wr * 128 + fr) * 128 + ((fq ^ frs) << 4));
  unsigned b_rd[4];
  {
    const int g = lane >> 4, q = (lane & 15) >> 2, p = lane & 3;
    const int k = 8 * g + q;
    const int h = (k & 1) | (((k >> 1) & 1) << 1) | (((k >> 3) & 1) << 2);
#pragma unroll
    for (int nf = 0; nf < 4; ++nf) b_rd[nf] = (unsigned)(B_BASE + k * 512 + (((wc * 4 + nf) ^ h) * 32) + p * 8);
  }
  const unsigned b_wr0 = (unsigned)(B_BASE + wid * 8 * 512 + (lane & 3) * 8);
  const unsigned b_wr1 = (unsigned)(((lane >> 2) ^ ((wid & 1) << 2)) * 32);
  const unsigned b_voff = (unsigned)(lane * 16);
  const unsigned a_dst = (unsigned)(size_t)smem + (unsigned)(wid * 4096);

  f32x4 acc[8][4];
#pragma unroll
  for (int m = 0; m < 8; ++m)
#pragma unroll
    for (int n = 0; n < 4; ++n) acc[m][n] = (f32x4){0.f, 0.f, 0.f, 0.f};

  f32x4 br[8];
  f16x8 At[4][2], B0[2][2], B1[2][2];

  const char* a_src = ws + WS_ASTAGE + parity * ASTAGE_BYTES + ((size_t)mt * KT_ALL) * IMG_BYTES + wid * 4096 + lane * 16;
  const char* w_src = (const char*)(W + (size_t)(wid * 8) * NN + (size_t)nt * 256);
  const unsigned* flag_base = (const unsigned*)(ws + WS_FLAGS) + mt * 32;

  const int ord_v = (int)g_order.v[kh][lane & 31];
  auto ord_of = [&](int t) -> int {
    t = t < KT_HALF ? t : KT_HALF - 1;
    return __builtin_amdgcn_readlane(ord_v, t >> 1);
  };
  auto kt_of = [&](int t) -> int {
    const int e = ord_of(t);
    return (((rank + (e >> 3)) & (NR - 1)) * 16) + (e & 7) * 2 + ((t < KT_HALF ? t : KT_HALF - 1) & 1);
  };
  auto flag_of = [&](int t) -> const unsigned* {
    const int e = ord_of(t);
    return flag_base + (size_t)(((rank + (e >> 3)) & (NR - 1)) * 8 + (e & 7)) * 128;
  };
  unsigned flv = 0;
  const unsigned vzero = 0;
#define FLAG_FETCH(t) do { const char* fp_ = uni_ptr(flag_of(t)); \
    asm volatile("global_load_dword %0, %1, %2 sc0 sc1" : "=v"(flv) : "v"(vzero), "s"(fp_)); } while (0)
#define FLAG_CHECK(t) do { asm volatile("" : "+v"(flv) :: "memory"); \
    if (__builtin_amdgcn_readfirstlane(flv) < call) bounded_wait(flag_of(t), call, tmo, call); } while (0)

#define ISSUE_A(kt, aoff, q) glds16(a_src + (size_t)(kt) * IMG_BYTES + (q) * 1024, (aoff) + a_dst + (q) * 1024)

#define ISSUE_B(kt, q) do { \
    const char* g_ = uni_ptr(w_src + (size_t)(kt) * (BK * (size_t)NN * 4)); \
    _Pragma("unroll") for (int j_ = 2 * (q); j_ < 2 * (q) + 2; ++j_) { \
      const char* gj_ = g_ + (size_t)j_ * (NN * 4); \
      asm volatile("global_load_dwordx4 %0, %1, %2" : "=v"(br[j_]) : "v"(b_voff), "s"(gj_)); \
    } } while (0)

#define WAIT_B(q, n) asm volatile("s_waitcnt vmcnt(" #n ")" : "+v"(br[2 * (q)]), "+v"(br[2 * (q) + 1]))

#define WRITE_B(stage, q) do { \
    _Pragma("unroll") for (int j_ = 2 * (q); j_ < 2 * (q) + 2; ++j_) { \
      f16x4 h_; \
      _Pragma("unroll") for (int i_ = 0; i_ < 4; ++i_) h_[i_] = (f16)br[j_][i_]; \
      *(f16x4*)(smem + (stage) * IMG_BYTES + j_ * 512 + b_wr0 + (b_wr1 ^ ((j_ & 3) * 32))) = h_; \
    } } while (0)

#define LDA(mh) do { \
    _Pragma("unroll") for (int m_ = 0; m_ < 4; ++m_) { \
      At[m_][0] = *(const f16x8*)(smem + a_rdc + ((mh) * 4 + m_) * 2048); \
      At[m_][1] = *(const f16x8*)(smem + (a_rdc ^ 64) + ((mh) * 4 + m_) * 2048); \
    } } while (0)
#define TRRD(off) __builtin_bit_cast(f16x4, __builtin_amdgcn_ds_read_tr16_b64_v4i16((lds_s4p)(smem + (off))))
#define LDB(dst, stage, nh) do { \
    _Pragma("unroll") for (int n_ = 0; n_ < 2; ++n_) \
    _Pragma("unroll") for (int k_ = 0; k_ < 2; ++k_) { \
      f16x4 lo_ = TRRD((stage) * IMG_BYTES + b_rd[(nh) * 2 + n_] + k_ * 16384); \
      f16x4 hi_ = TRRD((stage) * IMG_BYTES + b_rd[(nh) * 2 + n_] + k_ * 16384 + 2048); \
      dst[n_][k_] = __builtin_shufflevector(lo_, hi_, 0, 1, 2, 3, 4, 5, 6, 7); \
    } } while (0)
#define MMA(mh, nh, Bf) do { __builtin_amdgcn_s_setprio(1); \
    _Pragma("unroll") for (int m_ = 0; m_ < 4; ++m_) \
    _Pragma("unroll") for (int n_ = 0; n_ < 2; ++n_) \
    _Pragma("unroll") for (int k_ = 0; k_ < 2; ++k_) \
      acc[(mh) * 4 + m_][(nh) * 2 + n_] = __builtin_amdgcn_mfma_f32_16x16x32_f16(Bf[n_][k_], At[m_][k_], acc[(mh) * 4 + m_][(nh) * 2 + n_], 0, 0, 0); \
    __builtin_amdgcn_s_setprio(0); } while (0)

  {
    const int k0 = kt_of(0), k1 = kt_of(1);
    ISSUE_B(k0, 0); ISSUE_B(k0, 1); ISSUE_B(k0, 2); ISSUE_B(k0, 3);
    bounded_wait(flag_of(0), call, tmo, call);
    FLAG_FETCH(2);
    ISSUE_A(k0, 0, 0); ISSUE_A(k0, 0, 1); ISSUE_A(k0, 0, 2); ISSUE_A(k0, 0, 3);
    WAIT_B(0, 0); WRITE_B(0, 0);
    WAIT_B(1, 0); WRITE_B(0, 1);
    WAIT_B(2, 0); WRITE_B(0, 2);
    WAIT_B(3, 0); WRITE_B(0, 3);
    ISSUE_A(k1, IMG_BYTES, 0); ISSUE_B(k1, 0);
    ISSUE_A(k1, IMG_BYTES, 1); ISSUE_B(k1, 1);
    ISSUE_A(k1, IMG_BYTES, 2); ISSUE_B(k1, 2);
    ISSUE_A(k1, IMG_BYTES, 3); ISSUE_B(k1, 3);
  }
  WAIT_L(0);
  BAR();
  STAMP(2);
  if (wr == 1) BAR();

  unsigned a_rdc = a_rd;
  int a_cur = 0;
#define KTILE(t, s) do { \
    if (((t) & 1) == 0) { \
      if ((t) == t_push) do_push(); \
      if ((t) == t_push + 2) signal_push(); \
      FLAG_CHECK((t) + 2); \
      FLAG_FETCH((t) + 4); \
    } \
    const int ktn2 = kt_of((t) + 2); \
    const unsigned a_nx = (unsigned)(a_cur == 0 ? 2 : a_cur - 1) * IMG_BYTES; \
      \
    LDB(B0, s, 0); SCHED(); LDA(0); ISSUE_A(ktn2, a_nx, 0); \
    WAIT_B(0, 10); WRITE_B((s) ^ 1, 0); ISSUE_B(ktn2, 0); \
    BAR(); WAIT_L(0); MMA(0, 0, B0); BAR(); SCHED(); \
      \
    LDB(B1, s, 1); ISSUE_A(ktn2, a_nx, 1); \
    WAIT_B(1, 10); WRITE_B((s) ^ 1, 1); ISSUE_B(ktn2, 1); \
    BAR(); WAIT_L(0); MMA(0, 1, B1); BAR(); SCHED(); \
      \
    LDA(1); ISSUE_A(ktn2, a_nx, 2); \
    WAIT_B(2, 10); WRITE_B((s) ^ 1, 2); ISSUE_B(ktn2, 2); \
    BAR(); WAIT_L(0); MMA(1, 1, B1); BAR(); SCHED(); \
      \
    ISSUE_A(ktn2, a_nx, 3); \
    WAIT_B(3, 10); WRITE_B((s) ^ 1, 3); ISSUE_B(ktn2, 3); WAIT_L(0); \
    BAR(); MMA(1, 0, B0); \
    a_cur = (a_cur == 2) ? 0 : a_cur + 1; a_rdc = a_rd + (unsigned)a_cur * IMG_BYTES; \
    BAR(); SCHED(); \
  } while (0)

  for (int t = 0; t < KT_HALF; t += 2) {
    KTILE(t, 0);
    KTILE(t + 1, 1);
  }
#undef KTILE
  if (wr == 0) BAR();
  WAIT_V(0);
  BAR();
  STAMP(3);

  int tz = threadIdx.x; asm volatile("" : "+v"(tz));
  const int lane2 = tz & 63;
  const int fr2 = lane2 & 15, fq2 = lane2 >> 4;
  const int tile = nt * 4 + mt;
  unsigned* const sk = (unsigned*)(ws + WS_SPLITK + (size_t)tile * 256);
  const char* pbase = uni_ptr((f32x4*)(ws + WS_PARTIAL) + (size_t)tile * (32 * 512));
  {
    const char* pb = uni_ptr(pbase + (size_t)(kh ^ 1) * (16 * 8192));
    const unsigned voff = (unsigned)tz * 16u;
    if (kh == 0) {
#pragma unroll
      for (int m = 4; m < 8; ++m)
#pragma unroll
        for (int n = 0; n < 4; ++n) {
          const char* pbi = pb + (size_t)((m - 4) * 4 + n) * 8192;
          asm volatile("global_store_dwordx4 %0, %1, %2 sc1\n\ts_nop 1" :: "v"(voff), "v"(acc[m][n]), "s"(pbi) : "memory");
        }
    } else {
#pragma unroll
      for (int m = 0; m < 4; ++m)
#pragma unroll
        for (int n = 0; n < 4; ++n) {
          const char* pbi = pb + (size_t)(m * 4 + n) * 8192;
          asm volatile("global_store_dwordx4 %0, %1, %2 sc1\n\ts_nop 1" :: "v"(voff), "v"(acc[m][n]), "s"(pbi) : "memory");
        }
    }
  }
  WAIT_V(0);
  __syncthreads();
  volatile unsigned* s_tk = (volatile unsigned*)smem;
  if (tz == 0) {
    __hip_atomic_store(sk + 32 * kh, call, __ATOMIC_RELAXED, __HIP_MEMORY_SCOPE_AGENT);
    STAMP(4);
    bounded_wait(sk + 32 * (kh ^ 1), call, tmo, call);
    __builtin_amdgcn_fence(__ATOMIC_ACQUIRE, "agent");
    asm volatile("s_waitcnt vmcnt(0)" ::: "memory");
    *s_tk = (__hip_atomic_load(tmo, __ATOMIC_RELAXED, __HIP_MEMORY_SCOPE_AGENT) == call) ? 1u : 0u;
  }
  __syncthreads();
  const bool poisoned = __builtin_amdgcn_readfirstlane(*s_tk) != 0u;
  STAMP(5);
  float* orow = out + (size_t)(mt * 256 + wr * 128 + kh * 64 + fr2) * NN + nt * 256 + wc * 64 + fq2 * 4;
  if (poisoned) {
    const float qn = __builtin_nanf("");
#pragma unroll
    for (int m = 0; m < 4; ++m)
#pragma unroll
      for (int n = 0; n < 4; ++n) *(f32x4*)(orow + (size_t)m * 16 * NN + n * 16) = (f32x4){qn, qn, qn, qn};
    return;
  }
  const f32x4* pp = (const f32x4*)(pbase + (size_t)kh * (16 * 8192)) + tz;
  if (kh == 0) {
#pragma unroll
    for (int m = 0; m < 4; ++m)
#pragma unroll
      for (int n = 0; n < 4; ++n) {
        f32x4 v = acc[m][n] + pp[(m * 4 + n) * 512];
        v[0] = fmaxf(v[0], 0.f); v[1] = fmaxf(v[1], 0.f); v[2] = fmaxf(v[2], 0.f); v[3] = fmaxf(v[3], 0.f);
        *(f32x4*)(orow + (size_t)m * 16 * NN + n * 16) = v;
      }
  } else {
#pragma unroll
    for (int m = 0; m < 4; ++m)
#pragma unroll
      for (int n = 0; n < 4; ++n) {
        f32x4 v = acc[m + 4][n] + pp[(m * 4 + n) * 512];
        v[0] = fmaxf(v[0], 0.f); v[1] = fmaxf(v[1], 0.f); v[2] = fmaxf(v[2], 0.f); v[3] = fmaxf(v[3], 0.f);
        *(f32x4*)(orow + (size_t)m * 16 * NN + n * 16) = v;
      }
  }
}

extern "C" void kernel_launch(int rank, int nranks, ncclComm_t comm,
                              hipStream_t stream,
                              void* const* d_in, const int* in_sizes, int n_in,
                              void* d_out, int out_size,
                              const int* in_dtypes, int out_dtype,
                              void* const* d_out_all,
                              void* const* d_ws_all, size_t ws_bytes) {
  (void)comm; (void)in_sizes; (void)n_in; (void)out_size; (void)in_dtypes; (void)out_dtype; (void)d_out_all;
  if (nranks != NR || (unsigned)rank >= (unsigned)NR || ws_bytes < WS_TOTAL) {
    fprintf(stderr, "a2a_gemm: unexpected configuration\n");
    return;
  }
  static bool inited = false;
  if (!inited) {
    HIP_CHECK(hipFuncSetAttribute((const void*)a2a_gemm_kernel, hipFuncAttributeMaxDynamicSharedMemorySize, LDS_BYTES));
    inited = true;
  }
  PeerTable pt;
  for (int p = 0; p < NR; ++p) pt.ws[p] = (char*)d_ws_all[p];
  hipLaunchKernelGGL(a2a_gemm_kernel, dim3(GRID), dim3(512), LDS_BYTES, stream,
                     (const float*)d_in[0], (const float*)d_in[1], (float*)d_out, pt, rank);
  HIP_CHECK(hipGetLastError());
}
```

```cpp
#include <hip/hip_runtime.h>
#include <rccl/rccl.h>
#include <cstdio>
#include <cstdint>

#define HIP_CHECK(expr) do { hipError_t _e = (expr); if (_e != hipSuccess) { \
    fprintf(stderr, "HIP error %s at %s:%d\n", hipGetErrorString(_e), __FILE__, __LINE__); \
    return; } } while (0)

typedef _Float16 f16;
typedef f16 f16x8 __attribute__((ext_vector_type(8)));
typedef f16 f16x4 __attribute__((ext_vector_type(4)));
typedef float f32x4 __attribute__((ext_vector_type(4)));
typedef short s16x4v __attribute__((__vector_size__(4 * sizeof(short))));
typedef __attribute__((address_space(3))) s16x4v* lds_s4p;

namespace {
constexpr int NR = 8;
constexpr int ML = 1024, NN = 8192, KD = 8192, KL = 1024;
constexpr int BK = 64;
constexpr int KT_ALL = KD / BK;
constexpr int KT_HALF = KT_ALL / 2;
constexpr int IMG_BYTES = 256 * BK * 2;
constexpr int LDS_BYTES = 5 * IMG_BYTES;
constexpr int B_BASE = 3 * IMG_BYTES;
constexpr int GRID = 256;

constexpr size_t WS_TICKET = 0;
constexpr size_t WS_TMO = 128;
constexpr size_t WS_FLAGS = 4096;
constexpr size_t WS_SPLITK = 65536;
constexpr size_t WS_ASTAGE = 1u << 20;
constexpr size_t ASTAGE_BYTES = (size_t)4 * KT_ALL * IMG_BYTES;
constexpr size_t WS_PARTIAL = WS_ASTAGE + 2 * ASTAGE_BYTES;
constexpr size_t WS_TOTAL = WS_PARTIAL + (size_t)128 * 32 * 512 * 16;

constexpr unsigned long long DEADLINE_TICKS = 20000000ull;

struct PeerTable { char* ws[NR]; };

struct OrderTab { unsigned char v[2][32]; };
constexpr OrderTab make_order() {
  OrderTab t{};
  for (int h = 0; h < 2; ++h) {
    int n = 0;
    for (int kc = 4 * h; kc < 4 * h + 4; ++kc) t.v[h][n++] = (unsigned char)kc;
    for (int kc = 0; kc < 8; ++kc)
      for (int sr = 1; sr < 8; ++sr)
        if (((sr + kc) & 1) == h) t.v[h][n++] = (unsigned char)(sr * 8 + kc);
  }
  return t;
}
__constant__ OrderTab g_order = make_order();
}

__device__ __forceinline__ void glds16(const void* gsrc, unsigned lds_dst) {
  unsigned keep;
  asm volatile("s_mov_b32 %0, m0\n\ts_mov_b32 m0, %2\n\ts_nop 0\n\tglobal_load_lds_dwordx4 %1, off\n\ts_mov_b32 m0, %0"
               : "=&s"(keep) : "v"(gsrc), "s"(lds_dst) : "memory");
}

#define WAIT_V(n) asm volatile("s_waitcnt vmcnt(" #n ")" ::: "memory")
#define WAIT_L(n) asm volatile("s_waitcnt lgkmcnt(" #n ")" ::: "memory")
#define BAR() do { asm volatile("" ::: "memory"); __builtin_amdgcn_s_barrier(); asm volatile("" ::: "memory"); } while (0)
#define SCHED() __builtin_amdgcn_sched_barrier(0)

__device__ __forceinline__ void bounded_wait(const unsigned* flag, unsigned want, unsigned* tmo, unsigned call) {
  const unsigned long long t0 = __builtin_amdgcn_s_memrealtime();
  while (__hip_atomic_load(flag, __ATOMIC_RELAXED, __HIP_MEMORY_SCOPE_SYSTEM) < want) {
    if (__hip_atomic_load(tmo, __ATOMIC_RELAXED, __HIP_MEMORY_SCOPE_AGENT) == call ||
        __builtin_amdgcn_s_memrealtime() - t0 > DEADLINE_TICKS) {
      __hip_atomic_store(tmo, call, __ATOMIC_RELAXED, __HIP_MEMORY_SCOPE_AGENT);
      break;
    }
    __builtin_amdgcn_s_sleep(2);
  }
}

__device__ __forceinline__ const char* uni_ptr(const void* p) {
  const unsigned long long v = (unsigned long long)p;
  unsigned lo = __builtin_amdgcn_readfirstlane((unsigned)v), hi = __builtin_amdgcn_readfirstlane((unsigned)(v >> 32));
  asm volatile("s_nop 4" : "+s"(lo), "+s"(hi));
  return (const char*)(((unsigned long long)hi << 32) | lo);
}

__global__ __launch_bounds__(512, 2)
void a2a_gemm_kernel(const float* __restrict__ X,
                     const float* __restrict__ W,
                     float* __restrict__ out,
                     PeerTable peers, int rank)
{
  extern __shared__ __attribute__((aligned(16))) char smem[];
  char* const ws = peers.ws[rank];
  unsigned* const tmo = (unsigned*)(ws + WS_TMO);
  const int tid = threadIdx.x;
  const int lane = tid & 63;
  const int wid = __builtin_amdgcn_readfirstlane(tid >> 6);
  const int wr = wid >> 2, wc = wid & 3;
  const int fr = lane & 15, fq = lane >> 4;
  const int b = blockIdx.x;

  const int xcd = b & 7, idx = b >> 3;
  const int nt = xcd * 4 + (idx & 3);
  const int mt = (idx >> 2) & 3;
  const int kh = idx >> 4;

  const int frs = ((fr >> 1) & 7) ^ ((fr & 1) << 2);
  const unsigned a_rd = (unsigned)((wr * 128 + fr) * 128 + ((fq ^ frs) << 4));
  unsigned b_rd[4];
  {
    const int g = lane >> 4, q = (lane & 15) >> 2, p = lane & 3;
    const int k = 8 * g + q;
    const int h = (k & 1) | (((k >> 1) & 1) << 1) | (((k >> 3) & 1) << 2);
#pragma unroll
    for (int nf = 0; nf < 4; ++nf) b_rd[nf] = (unsigned)(B_BASE + k * 512 + (((wc * 4 + nf) ^ h) * 32) + p * 8);
  }
  const unsigned b_wr0 = (unsigned)(B_BASE + wid * 8 * 512 + (lane & 3) * 8);
  const unsigned b_wr1 = (unsigned)(((lane >> 2) ^ ((wid & 1) << 2)) * 32);
  const unsigned b_voff = (unsigned)(lane * 16);
  const unsigned a_dst = (unsigned)(size_t)smem + (unsigned)(wid * 4096);

  f32x4 acc[8][4];
#pragma unroll
  for (int m = 0; m < 8; ++m)
#pragma unroll
    for (int n = 0; n < 4; ++n) acc[m][n] = (f32x4){0.f, 0.f, 0.f, 0.f};

  f32x4 br[8];
  f16x8 At[4][2], B0[2][2], B1[2][2];

  const char* w_src = (const char*)(W + (size_t)(wid * 8) * NN + (size_t)nt * 256);
  const unsigned* flag_base = (const unsigned*)(ws + WS_FLAGS) + mt * 32;

  const int ord_v = (int)g_order.v[kh][lane & 31];
  auto ord_of = [&](int t) -> int {
    t = t < KT_HALF ? t : KT_HALF - 1;
    return __builtin_amdgcn_readlane(ord_v, t >> 1);
  };
  auto kt_of = [&](int t) -> int {
    const int e = ord_of(t);
    return (((rank + (e >> 3)) & (NR - 1)) * 16) + (e & 7) * 2 + ((t < KT_HALF ? t : KT_HALF - 1) & 1);
  };
  auto flag_of = [&](int t) -> const unsigned* {
    const int e = ord_of(t);
    return flag_base + (size_t)(((rank + (e >> 3)) & (NR - 1)) * 8 + (e & 7)) * 128;
  };
  unsigned flv = 0;
  const unsigned vzero = 0;
#define FLAG_FETCH(t) do { const char* fp_ = uni_ptr(flag_of(t)); \
    asm volatile("global_load_dword %0, %1, %2 sc0 sc1" : "=v"(flv) : "v"(vzero), "s"(fp_)); } while (0)
#define FLAG_CHECK(t) do { asm volatile("" : "+v"(flv) :: "memory"); \
    if (__builtin_amdgcn_readfirstlane(flv) < call) bounded_wait(flag_of(t), call, tmo, call); } while (0)

#define ISSUE_A(kt, aoff, q) glds16(a_src + (size_t)(kt) * IMG_BYTES + (q) * 1024, (aoff) + a_dst + (q) * 1024)

#define ISSUE_B(kt, q) do { \
    const char* g_ = uni_ptr(w_src + (size_t)(kt) * (BK * (size_t)NN * 4)); \
    _Pragma("unroll") for (int j_ = 2 * (q); j_ < 2 * (q) + 2; ++j_) { \
      const char* gj_ = g_ + (size_t)j_ * (NN * 4); \
      asm volatile("global_load_dwordx4 %0, %1, %2" : "=v"(br[j_]) : "v"(b_voff), "s"(gj_)); \
    } } while (0)

#define WAIT_B(q, n) asm volatile("s_waitcnt vmcnt(" #n ")" : "+v"(br[2 * (q)]), "+v"(br[2 * (q) + 1]))

#define WRITE_B(stage, q) do { \
    _Pragma("unroll") for (int j_ = 2 * (q); j_ < 2 * (q) + 2; ++j_) { \
      f16x4 h_; \
      _Pragma("unroll") for (int i_ = 0; i_ < 4; ++i_) h_[i_] = (f16)br[j_][i_]; \
      *(f16x4*)(smem + (stage) * IMG_BYTES + j_ * 512 + b_wr0 + (b_wr1 ^ ((j_ & 3) * 32))) = h_; \
    } } while (0)

#define LDA(mh) do { \
    _Pragma("unroll") for (int m_ = 0; m_ < 4; ++m_) { \
      At[m_][0] = *(const f16x8*)(smem + a_rdc + ((mh) * 4 + m_) * 2048); \
      At[m_][1] = *(const f16x8*)(smem + (a_rdc ^ 64) + ((mh) * 4 + m_) * 2048); \
    } } while (0)
#define TRRD(off) __builtin_bit_cast(f16x4, __builtin_amdgcn_ds_read_tr16_b64_v4i16((lds_s4p)(smem + (off))))
#define LDB(dst, stage, nh) do { \
    _Pragma("unroll") for (int n_ = 0; n_ < 2; ++n_) \
    _Pragma("unroll") for (int k_ = 0; k_ < 2; ++k_) { \
      f16x4 lo_ = TRRD((stage) * IMG_BYTES + b_rd[(nh) * 2 + n_] + k_ * 16384); \
      f16x4 hi_ = TRRD((stage) * IMG_BYTES + b_rd[(nh) * 2 + n_] + k_ * 16384 + 2048); \
      dst[n_][k_] = __builtin_shufflevector(lo_, hi_, 0, 1, 2, 3, 4, 5, 6, 7); \
    } } while (0)
#define MMA(mh, nh, Bf) do { __builtin_amdgcn_s_setprio(1); \
    _Pragma("unroll") for (int m_ = 0; m_ < 4; ++m_) \
    _Pragma("unroll") for (int n_ = 0; n_ < 2; ++n_) \
    _Pragma("unroll") for (int k_ = 0; k_ < 2; ++k_) \
      acc[(mh) * 4 + m_][(nh) * 2 + n_] = __builtin_amdgcn_mfma_f32_16x16x32_f16(Bf[n_][k_], At[m_][k_], acc[(mh) * 4 + m_][(nh) * 2 + n_], 0, 0, 0); \
    __builtin_amdgcn_s_setprio(0); } while (0)

  const int k0 = kt_of(0), k1 = kt_of(1);
  ISSUE_B(k0, 0); ISSUE_B(k0, 1); ISSUE_B(k0, 2); ISSUE_B(k0, 3);

  volatile unsigned* s_bc = (volatile unsigned*)(smem + 2 * IMG_BYTES);
  if (tid == 0) {
    unsigned tk = __hip_atomic_fetch_add((unsigned*)(ws + WS_TICKET), 1u, __ATOMIC_RELAXED, __HIP_MEMORY_SCOPE_AGENT);
    *s_bc = tk / GRID + 1u;
  }
  __syncthreads();
  const unsigned call = __builtin_amdgcn_readfirstlane(*s_bc);
  const unsigned parity = call & 1u;

  const int p_kc = b >> 5, p_drel = (b >> 2) & 7, p_mt = b & 3;
  const int p_dst = (rank + p_drel) & (NR - 1);
  const int t_push = (p_drel == 0) ? -1 : 4 * p_kc;
  auto do_push = [&]() {
    const char* dbase = uni_ptr(peers.ws[p_dst] + WS_ASTAGE + parity * ASTAGE_BYTES +
                                ((size_t)p_mt * KT_ALL + (size_t)(rank * 16 + p_kc * 2)) * IMG_BYTES);
    const float* xs = X + (size_t)(p_dst * ML + p_mt * 256) * KL + p_kc * 128;
    int tq = threadIdx.x; asm volatile("" : "+v"(tq));
    f32x4 v[8][2];
#pragma unroll
    for (int i = 0; i < 8; ++i) {
      const int c = i * 512 + tq;
      const int img = c >> 11, pos = c & 2047, R = pos >> 3;
      const int ch = (pos & 7) ^ (((R >> 1) & 7) ^ ((R & 1) << 2));
      const float* p = xs + (size_t)R * KL + img * 64 + ch * 8;
      v[i][0] = *(const f32x4*)p;
      v[i][1] = *(const f32x4*)(p + 4);
    }
    const unsigned voff = (unsigned)tq * 16u;
#pragma unroll
    for (int i = 0; i < 8; ++i) {
      f16x8 h;
#pragma unroll
      for (int e = 0; e < 4; ++e) { h[e] = (f16)v[i][0][e]; h[4 + e] = (f16)v[i][1][e]; }
      const char* db = dbase + (size_t)i * 8192;
      asm volatile("global_store_dwordx4 %0, %1, %2 sc0 sc1\n\ts_nop 1" :: "v"(voff), "v"(h), "s"(db) : "memory");
    }
  };
  auto signal_push = [&]() {
    if (tid == 0)
      __hip_atomic_store((unsigned*)(peers.ws[p_dst] + WS_FLAGS + (size_t)((rank * 8 + p_kc) * 4 + p_mt) * 128), call,
                         __ATOMIC_RELAXED, __HIP_MEMORY_SCOPE_SYSTEM);
  };
  if (t_push < 0) {
    do_push();
    WAIT_V(0);
    __syncthreads();
    signal_push();
  }

  const char* a_src = ws + WS_ASTAGE + parity * ASTAGE_BYTES + ((size_t)mt * KT_ALL) * IMG_BYTES + wid * 4096 + lane * 16;

  {
    bounded_wait(flag_of(0), call, tmo, call);
    FLAG_FETCH(2);
    ISSUE_A(k0, 0, 0); ISSUE_A(k0, 0, 1); ISSUE_A(k0, 0, 2); ISSUE_A(k0, 0, 3);
    WAIT_B(0, 0); WRITE_B(0, 0);
    WAIT_B(1, 0); WRITE_B(0, 1);
    WAIT_B(2, 0); WRITE_B(0, 2);
    WAIT_B(3, 0); WRITE_B(0, 3);
    ISSUE_A(k1, IMG_BYTES, 0); ISSUE_B(k1, 0);
    ISSUE_A(k1, IMG_BYTES, 1); ISSUE_B(k1, 1);
    ISSUE_A(k1, IMG_BYTES, 2); ISSUE_B(k1, 2);
    ISSUE_A(k1, IMG_BYTES, 3); ISSUE_B(k1, 3);
  }
  WAIT_L(0);
  BAR();
  if (wr == 1) BAR();

  unsigned a_rdc = a_rd;
  int a_cur = 0;
#define KTILE(t, s) do { \
    if (((t) & 1) == 0) { \
      if ((t) == t_push) do_push(); \
      if ((t) == t_push + 2) signal_push(); \
      FLAG_CHECK((t) + 2); \
      FLAG_FETCH((t) + 4); \
    } \
    const int ktn2 = kt_of((t) + 2); \
    const unsigned a_nx = (unsigned)(a_cur == 0 ? 2 : a_cur - 1) * IMG_BYTES; \
      \
    LDB(B0, s, 0); SCHED(); LDA(0); ISSUE_A(ktn2, a_nx, 0); \
    WAIT_B(0, 10); WRITE_B((s) ^ 1, 0); ISSUE_B(ktn2, 0); \
    BAR(); WAIT_L(0); MMA(0, 0, B0); BAR(); SCHED(); \
      \
    LDB(B1, s, 1); ISSUE_A(ktn2, a_nx, 1); \
    WAIT_B(1, 10); WRITE_B((s) ^ 1, 1); ISSUE_B(ktn2, 1); \
    BAR(); WAIT_L(0); MMA(0, 1, B1); BAR(); SCHED(); \
      \
    LDA(1); ISSUE_A(ktn2, a_nx, 2); \
    WAIT_B(2, 10); WRITE_B((s) ^ 1, 2); ISSUE_B(ktn2, 2); \
    BAR(); WAIT_L(0); MMA(1, 1, B1); BAR(); SCHED(); \
      \
    ISSUE_A(ktn2, a_nx, 3); \
    WAIT_B(3, 10); WRITE_B((s) ^ 1, 3); ISSUE_B(ktn2, 3); WAIT_L(0); \
    BAR(); MMA(1, 0, B0); \
    a_cur = (a_cur == 2) ? 0 : a_cur + 1; a_rdc = a_rd + (unsigned)a_cur * IMG_BYTES; \
    BAR(); SCHED(); \
  } while (0)

  for (int t = 0; t < KT_HALF; t += 2) {
    KTILE(t, 0);
    KTILE(t + 1, 1);
  }
#undef KTILE
  if (wr == 0) BAR();
  WAIT_V(0);
  BAR();

  int tz = threadIdx.x; asm volatile("" : "+v"(tz));
  const int lane2 = tz & 63;
  const int fr2 = lane2 & 15, fq2 = lane2 >> 4;
  const int tile = nt * 4 + mt;
  unsigned* const sk = (unsigned*)(ws + WS_SPLITK + (size_t)tile * 256);
  const char* pbase = uni_ptr((f32x4*)(ws + WS_PARTIAL) + (size_t)tile * (32 * 512));
  {
    const char* pb = uni_ptr(pbase + (size_t)(kh ^ 1) * (16 * 8192));
    const unsigned voff = (unsigned)tz * 16u;
    if (kh == 0) {
#pragma unroll
      for (int m = 4; m < 8; ++m)
#pragma unroll
        for (int n = 0; n < 4; ++n) {
          const char* pbi = pb + (size_t)((m - 4) * 4 + n) * 8192;
          asm volatile("global_store_dwordx4 %0, %1, %2 sc1\n\ts_nop 1" :: "v"(voff), "v"(acc[m][n]), "s"(pbi) : "memory");
        }
    } else {
#pragma unroll
      for (int m = 0; m < 4; ++m)
#pragma unroll
        for (int n = 0; n < 4; ++n) {
          const char* pbi = pb + (size_t)(m * 4 + n) * 8192;
          asm volatile("global_store_dwordx4 %0, %1, %2 sc1\n\ts_nop 1" :: "v"(voff), "v"(acc[m][n]), "s"(pbi) : "memory");
        }
    }
  }
  WAIT_V(0);
  __syncthreads();
  volatile unsigned* s_tk = (volatile unsigned*)smem;
  if (tz == 0) {
    __hip_atomic_store(sk + 32 * kh, call, __ATOMIC_RELAXED, __HIP_MEMORY_SCOPE_AGENT);
    bounded_wait(sk + 32 * (kh ^ 1), call, tmo, call);
    __builtin_amdgcn_fence(__ATOMIC_ACQUIRE, "agent");
    asm volatile("s_waitcnt vmcnt(0)" ::: "memory");
    *s_tk = (__hip_atomic_load(tmo, __ATOMIC_RELAXED, __HIP_MEMORY_SCOPE_AGENT) == call) ? 1u : 0u;
  }
  __syncthreads();
  const bool poisoned = __builtin_amdgcn_readfirstlane(*s_tk) != 0u;
  float* orow = out + (size_t)(mt * 256 + wr * 128 + kh * 64 + fr2) * NN + nt * 256 + wc * 64 + fq2 * 4;
  if (poisoned) {
    const float qn = __builtin_nanf("");
#pragma unroll
    for (int m = 0; m < 4; ++m)
#pragma unroll
      for (int n = 0; n < 4; ++n) *(f32x4*)(orow + (size_t)m * 16 * NN + n * 16) = (f32x4){qn, qn, qn, qn};
    return;
  }
  const f32x4* pp = (const f32x4*)(pbase + (size_t)kh * (16 * 8192)) + tz;
  if (kh == 0) {
#pragma unroll
    for (int m = 0; m < 4; ++m)
#pragma unroll
      for (int n = 0; n < 4; ++n) {
        f32x4 v = acc[m][n] + pp[(m * 4 + n) * 512];
        v[0] = fmaxf(v[0], 0.f); v[1] = fmaxf(v[1], 0.f); v[2] = fmaxf(v[2], 0.f); v[3] = fmaxf(v[3], 0.f);
        *(f32x4*)(orow + (size_t)m * 16 * NN + n * 16) = v;
      }
  } else {
#pragma unroll
    for (int m = 0; m < 4; ++m)
#pragma unroll
      for (int n = 0; n < 4; ++n) {
        f32x4 v = acc[m + 4][n] + pp[(m * 4 + n) * 512];
        v[0] = fmaxf(v[0], 0.f); v[1] = fmaxf(v[1], 0.f); v[2] = fmaxf(v[2], 0.f); v[3] = fmaxf(v[3], 0.f);
        *(f32x4*)(orow + (size_t)m * 16 * NN + n * 16) = v;
      }
  }
}

extern "C" void kernel_launch(int rank, int nranks, ncclComm_t comm,
                              hipStream_t stream,
                              void* const* d_in, const int* in_sizes, int n_in,
                              void* d_out, int out_size,
                              const int* in_dtypes, int out_dtype,
                              void* const* d_out_all,
                              void* const* d_ws_all, size_t ws_bytes) {
  (void)comm; (void)in_sizes; (void)n_in; (void)out_size; (void)in_dtypes; (void)out_dtype; (void)d_out_all;
  if (nranks != NR || (unsigned)rank >= (unsigned)NR || ws_bytes < WS_TOTAL) {
    fprintf(stderr, "a2a_gemm: unexpected configuration\n");
    return;
  }
  static bool inited = false;
  if (!inited) {
    HIP_CHECK(hipFuncSetAttribute((const void*)a2a_gemm_kernel, hipFuncAttributeMaxDynamicSharedMemorySize, LDS_BYTES));
    inited = true;
  }
  PeerTable pt;
  for (int p = 0; p < NR; ++p) pt.ws[p] = (char*)d_ws_all[p];
  hipLaunchKernelGGL(a2a_gemm_kernel, dim3(GRID), dim3(512), LDS_BYTES, stream,
                     (const float*)d_in[0], (const float*)d_in[1], (float*)d_out, pt, rank);
  HIP_CHECK(hipGetLastError());
}
```
